```python
import numpy as np

kernel_name = "gpu_gemm_rs_m4096_k4096_n4096_bf16_gelu"


ABSMAX_RTOL = 0.008

COMPUTE_DTYPE = "bfloat16"
EPILOGUE = "gelu"
IN_QUANT = None
OUT_QUANT = None
IN_DTYPES = {"x": "bfloat16", "w_mat": "bfloat16"}
WORLD_SIZE = 8

def _to_bf16(x):
    u = np.ascontiguousarray(x, dtype=np.float32).view(np.uint32)
    return ((u + 0x7FFF + ((u >> 16) & 1)) & 0xFFFF0000).view(np.float32)

def setup_inputs():

    x = _to_bf16(np.random.randn(4096, 4096).astype(np.float32))
    w_mat = _to_bf16((np.random.randn(4096, 4096) / np.sqrt(4096)).astype(np.float32))
    return {"x": x, "w_mat": w_mat}

def _epilogue(y):
    c = 0.7978845608028654
    return (0.5 * y * (1.0 + np.tanh(c * (y + 0.044715 * y ** 3)))).astype(np.float32)

def reference(x, w_mat):
    y = (x @ w_mat).astype(np.float32)
    return _epilogue(y)
```

```cpp
#include <hip/hip_runtime.h>
#include <rccl/rccl.h>
#include <cstdio>
#include <cstdint>

#define HIP_CHECK(expr) do { hipError_t _e = (expr); if (_e != hipSuccess) { \
    fprintf(stderr, "HIP error %s at %s:%d\n", hipGetErrorString(_e), __FILE__, __LINE__); \
    return; } } while (0)

typedef unsigned int u32;
typedef unsigned long long u64;
typedef u32 u32x4 __attribute__((ext_vector_type(4)));
typedef u32 u32x2 __attribute__((ext_vector_type(2)));
typedef float f32x4 __attribute__((ext_vector_type(4)));
typedef __bf16 bf16x8 __attribute__((ext_vector_type(8)));
typedef __bf16 bf16x2 __attribute__((ext_vector_type(2)));
typedef float f32x2 __attribute__((ext_vector_type(2)));
#define GAS __attribute__((address_space(1)))

constexpr int NR = 8;
constexpr int NWG = 256;
constexpr int NTHREADS = 384;
#define PUSHERS 32
#define DEFER 4
#define DELTA_Q4 456
#define START_TICKS 110
#define HOLD_PRE 50
#define NEAR_TICKS 100
#define FROW 4352
#define POLL_SLEEP 8
#define MASTER_SLEEP 4
#define CPOLL_SLEEP 8
#define GEMM_AUX 0
#define OUT_AUX 0
#define LAYOUT 1
#define IDX_STRIDE 16384
#define STEP_STRIDE (8 * 16384)
constexpr size_t MIB = 1u << 20;
constexpr size_t XB_OFF = 0, WB_OFF = 8 * MIB, CB_OFF = 16 * MIB, PAR_BYTES = 24 * MIB;
constexpr size_t FLAG_OFF = 2 * PAR_BYTES;
constexpr size_t FX_OFF = FLAG_OFF;
constexpr size_t FW_OFF = FLAG_OFF + 8 * FROW;
constexpr size_t GATE_OFF = FLAG_OFF + 16 * FROW;
constexpr size_t PRIV_OFF = GATE_OFF + 8 * FROW;
constexpr size_t SEQ_OFF = PRIV_OFF + NWG * FROW;
constexpr size_t DBG_OFF = SEQ_OFF + NWG * 8 * 4;
constexpr size_t WS_NEED = DBG_OFF + NWG * 8 * 8 * 8;
constexpr u64 DEADLINE_TICKS = 40000000ull;

struct Params {
    const char* x;
    const char* w;
    char* out;
    char* ws[NR];
    int rank;
    u32 order[128];
};

typedef __amdgpu_buffer_rsrc_t rsrc_t;
__device__ __forceinline__ rsrc_t mk_rsrc(const void* p, size_t bytes) {
    return __builtin_amdgcn_make_buffer_rsrc((void*)p, (short)0, (int)bytes, 0x00020000);
}
constexpr int AUX_SYS = 17;
__device__ __forceinline__ void st16_sys(rsrc_t r, u32 voff, u32 soff, u32x4 v) {
    __builtin_amdgcn_raw_buffer_store_b128(v, r, voff, soff, AUX_SYS);
}
template <int AUX = 0>
__device__ __forceinline__ u32x4 ld16(rsrc_t r, u32 voff, u32 soff) {
    return __builtin_amdgcn_raw_buffer_load_b128(r, voff, soff, AUX);
}
__device__ __forceinline__ u32 ld_sys(const char* p) {
    return __hip_atomic_load((const GAS u32*)p, __ATOMIC_RELAXED, __HIP_MEMORY_SCOPE_SYSTEM);
}
__device__ __forceinline__ void st_sys(char* p, u32 v) {
    __hip_atomic_store((GAS u32*)p, v, __ATOMIC_RELAXED, __HIP_MEMORY_SCOPE_SYSTEM);
}
__device__ __forceinline__ u32 ld_agent(const char* p) {
    return __hip_atomic_load((const GAS u32*)p, __ATOMIC_RELAXED, __HIP_MEMORY_SCOPE_AGENT);
}
__device__ __forceinline__ void st_agent(char* p, u32 v) {
    __hip_atomic_store((GAS u32*)p, v, __ATOMIC_RELAXED, __HIP_MEMORY_SCOPE_AGENT);
}
#define WAIT_VM(n) asm volatile("s_waitcnt vmcnt(" #n ")" ::: "memory")

#define STAMP(i) do {} while (0)
#define STAMPD(i, d) do {} while (0)

struct __attribute__((aligned(16))) Shared {
    u32 ready;
    u32 hold;
    u32 gate;
    u32 slot;
    u32 near;
    u32 cready;
    u32 abort_;
    u32 cdone[2];
};

#define LAS __attribute__((address_space(3)))
__device__ __forceinline__ u32 lds_ld(u32* p) { return __builtin_amdgcn_readfirstlane(*(volatile LAS u32*)p); }
__device__ __forceinline__ void lds_st(u32* p, u32 v) { *(volatile LAS u32*)p = v; }

__device__ __forceinline__ void wait_until(u32 t) {
    while ((int)((u32)__builtin_amdgcn_s_memrealtime() - t) < 0) __builtin_amdgcn_s_sleep(1);
}

__device__ __forceinline__ u32 pack_bf16(float a, float b) {
    f32x2 v = {a, b};
    bf16x2 r = __builtin_convertvector(v, bf16x2);
    return __builtin_bit_cast(u32, r);
}
__device__ __forceinline__ float gelu_f(float y) {
    const float c2 = -2.0f * 0.7978845608028654f * 1.4426950408889634f;
    float u = y * (1.0f + 0.044715f * y * y);
    float e = __builtin_amdgcn_exp2f(c2 * u);
    return y * __builtin_amdgcn_rcpf(1.0f + e);
}
__device__ __forceinline__ float bf_lo(u32 v) { return __builtin_bit_cast(float, v << 16); }
__device__ __forceinline__ float bf_hi(u32 v) { return __builtin_bit_cast(float, v & 0xffff0000u); }

__device__ __forceinline__ void transpose8x8(u32x4 (&v)[8]) {
    u32x4 o[8];
#pragma unroll
    for (int c = 0; c < 4; ++c) {
#pragma unroll
        for (int r = 0; r < 4; ++r) {
            u32 a = v[2 * r][c], b = v[2 * r + 1][c];
            o[2 * c][r] = __builtin_amdgcn_perm(b, a, 0x05040100u);
            o[2 * c + 1][r] = __builtin_amdgcn_perm(b, a, 0x07060302u);
        }
    }
#pragma unroll
    for (int i = 0; i < 8; ++i) v[i] = o[i];
}

__global__ void __launch_bounds__(NTHREADS, 2) fused_gemm_rs(Params p) {
    __shared__ Shared sh;
    const int lane = threadIdx.x & 63;
    const int wave = __builtin_amdgcn_readfirstlane(threadIdx.x >> 6);
    const int grp = blockIdx.x & 7;
    const int tau = blockIdx.x >> 3;
    const int rank = p.rank;
    const int peer = (rank ^ grp) & 7;
    char* const myws = p.ws[rank & 7];
    STAMP(0);
    const u32 tstart = (u32)__builtin_amdgcn_s_memrealtime() + START_TICKS;

    if (wave >= 4) __builtin_amdgcn_s_setprio(3);
    char* const seqp = myws + SEQ_OFF + (size_t)(blockIdx.x * 8 + wave) * 4;
    u32 seq_raw = ld_agent(seqp);
    if (threadIdx.x == 0) { sh.ready = 0; sh.hold = 0; sh.gate = 0; sh.near = 0; sh.cready = 0; sh.abort_ = 0; sh.cdone[0] = 0; sh.cdone[1] = 0; }
    asm volatile("s_waitcnt lgkmcnt(0)" ::: "memory");

    if (wave == 4) {
        const int pw = tau - (32 - PUSHERS);
        const int pi = (peer >> 2) & 1, pj = (peer >> 1) & 1, pk = peer & 1;
        const int q = lane >> 4, r = lane & 15;
        const rsrc_t rsx = mk_rsrc(p.x, 4 * MIB), rsw = mk_rsrc(p.w, 4 * MIB);
        const u32 xv = (pi * 256 + r) * 1024 + pk * 512 + q * 16;
        const u32 wv = (pk * 256 + q * 8) * 8192 + pj * 4096 + r * 16;
        u32x4 va[8], vb[8];
        auto issue_loads = [&](int o) {
            const int ks = (o >> 3) & 7, idx = o & 7;
            if (o & 64) {
                const u32 so = ks * 32 * 8192 + idx * 512;
#pragma unroll
                for (int i = 0; i < 8; ++i) { va[i] = ld16(rsw, wv, so + i * 8192); vb[i] = ld16(rsw, wv + 256, so + i * 8192); }
            } else {
                const u32 so = ((peer ^ idx) & 7) * 512 * 1024 + ks * 64;
#pragma unroll
                for (int i = 0; i < 8; ++i) { va[i] = ld16(rsx, xv, so + i * 16384); vb[i] = ld16(rsx, xv, so + (i + 8) * 16384); }
            }
        };
        if (pw >= 0) issue_loads(p.order[pw]);
        __builtin_amdgcn_s_barrier();
        const u32 call = __builtin_amdgcn_readfirstlane(seq_raw) + 1;
        if (lane == 0) st_agent(seqp, call);
        if (pw < 0) return;
        char* const dws = p.ws[peer];
        const rsrc_t rsd = mk_rsrc(dws, FLAG_OFF);
        const u32 dpar = (call & 1) * (u32)PAR_BYTES;
        STAMP(1);
        for (int u = pw; u < 128; u += PUSHERS) {
            const int o = p.order[u];
            const int ks = (o >> 3) & 7, idx = o & 7;
            const bool isw = o & 64;
            const int n = grp == 0 ? ks : 8 + ks * 7 + (grp - 1);
            if (isw) { transpose8x8(va); transpose8x8(vb); }
            const u32 d = dpar + (u32)(isw ? WB_OFF : XB_OFF) + idx * IDX_STRIDE + n * STEP_STRIDE;
            if (grp != 0) {
                wait_until(tstart + ((u * DELTA_Q4) >> 4) - HOLD_PRE);
                if (lane == 0) lds_st(&sh.hold, 1);
                wait_until(tstart + ((u * DELTA_Q4) >> 4));
            }
#pragma unroll
            for (int i = 0; i < 8; ++i) st16_sys(rsd, lane * 16 + (i & 3) * 1024, d + (i >> 2) * 4096, va[i]);
#pragma unroll
            for (int i = 0; i < 8; ++i) st16_sys(rsd, lane * 16 + (i & 3) * 1024, d + 8192 + (i >> 2) * 4096, vb[i]);
            if (u + PUSHERS < 128) {
                issue_loads(p.order[u + PUSHERS]);
                WAIT_VM(16);
            } else {
                WAIT_VM(0);
            }
            if (lane == 0) { st_sys(dws + (isw ? FW_OFF : FX_OFF) + idx * FROW + n * 4, call); lds_st(&sh.hold, 0); }
        }
        STAMPD(7, dbg_late);
        STAMP(2);
        return;
    }

    __builtin_amdgcn_s_barrier();
    const u32 call = __builtin_amdgcn_readfirstlane(seq_raw) + 1;
    if (lane == 0) st_agent(seqp, call);
    const int rb = tau & 1, cb = tau >> 1;
    const int quad = grp >> 1;

    if (wave == 5) {
        WAIT_VM(0);
        const u64 t0 = __builtin_amdgcn_s_memrealtime();
        bool dead = false;
        const char* priv = myws + PRIV_OFF + (size_t)blockIdx.x * FROW + (lane & 7) * 4;
        auto expired = [&](u32 pv) -> bool {
            if (__builtin_amdgcn_readlane(pv, 4) == call) return true;
            if (__builtin_amdgcn_s_memrealtime() - t0 > DEADLINE_TICKS) {
                for (int i = 0; i < 4; ++i) st_agent(myws + PRIV_OFF + (size_t)(i * 64 + lane) * FROW + 16, call);
                return true;
            }
            return false;
        };
        bool master = grp == 0 && tau < 16;
        const char* frow = myws + FX_OFF + tau * FROW + lane * 4;
        const int cons = tau < 8 ? (lane & 31) * 8 + tau : (tau - 8) * 32 + (lane & 31);
        char* const pub_w = myws + PRIV_OFF + (size_t)cons * FROW + (tau >> 3) * 4;
        u32 pubd = 0, seen = 0;
        for (;;) {
            u32 a = 0;
            if (master) a = ld_sys(frow);
            const u32 pv = ld_sys(priv);
            if (master) {
                const u64 m = ~__ballot(a >= call);
                const u32 c = m ? (u32)__builtin_ctzll(m) : 64u;
                if (c != pubd) {
                    pubd = c;
                    if (lane < 32) st_agent(pub_w, call * 128 + c);
                    if (c == 64) master = false;
                }
            }
            const int c0 = (int)(__builtin_amdgcn_readlane(pv, 0) - call * 128), c1 = (int)(__builtin_amdgcn_readlane(pv, 1) - call * 128);
            const int c = min(c0, c1);
            if (c > (int)seen) {
                seen = c;
                if (lane == 0) lds_st(&sh.ready, seen);
            }
            if (seen == 64 && !master) break;
            if (expired(pv)) { dead = true; break; }
            if (master) __builtin_amdgcn_s_sleep(MASTER_SLEEP); else __builtin_amdgcn_s_sleep(POLL_SLEEP);
        }
        STAMP(1);
        if (dead && lane == 0) { lds_st(&sh.abort_, 1); lds_st(&sh.ready, 64); }
        while (!lds_ld(&sh.near)) __builtin_amdgcn_s_sleep(2);
        if (lane == 0) {
            const u64 now = __builtin_amdgcn_s_memrealtime();
            u64 slot = now;
            if (grp != 0) {
                u64 first = now + (u64)(long long)(int)(tstart + 8 * DELTA_Q4 - (u32)now);
                u64 lo = now + NEAR_TICKS;
                if ((long long)(first - lo) > 0) lo = first;
                GAS u64* clk = (GAS u64*)(myws + GATE_OFF + grp * FROW);
                __hip_atomic_fetch_max(clk, lo << 3, __ATOMIC_RELAXED, __HIP_MEMORY_SCOPE_AGENT);
                slot = __hip_atomic_fetch_add(clk, (u64)DELTA_Q4, __ATOMIC_RELAXED, __HIP_MEMORY_SCOPE_AGENT) >> 3;
                if (slot - now > 100000ull) slot = now;
            }
            lds_st(&sh.slot, (u32)slot);
            lds_st(&sh.gate, 1);
        }
        STAMP(2);
        if (!dead) {
            for (;;) {
                const u32 pv = ld_sys(priv);
                if (expired(pv)) { dead = true; break; }
                if (__builtin_amdgcn_readlane(pv, 2) >= call && __builtin_amdgcn_readlane(pv, 3) >= call) break;
                __builtin_amdgcn_s_sleep(CPOLL_SLEEP);
            }
        }
        if (dead && lane == 0) lds_st(&sh.abort_, 1);
        if (lane == 0) lds_st(&sh.cready, 1);
        STAMP(3);
        return;
    }

    const int wr = wave >> 1, wc = wave & 1;
    const rsrc_t rsm = mk_rsrc(myws, FLAG_OFF);
    const u32 par = (call & 1) * (u32)PAR_BYTES;
    const u32 xa = par + (u32)XB_OFF + grp * IDX_STRIDE + (rb * 8 + wr * 4) * 1024;
    const u32 wa = par + (u32)WB_OFF + (cb >> 1) * IDX_STRIDE + ((cb & 1) * 8 + wc * 4) * 1024;
    const u32 loff = lane * 16;
    f32x4 acc[4][4];
#pragma unroll
    for (int a = 0; a < 4; ++a)
#pragma unroll
        for (int b = 0; b < 4; ++b) acc[a][b] = f32x4{0.f, 0.f, 0.f, 0.f};
    bf16x8 fxr[4][4], fwr[4][4];
    u32x2 st = {0, 0};
    auto peek = [&]() { st = *(volatile LAS u32x2*)&sh.ready; };
    auto wait_step = [&](int n) {
        for (;;) {
            const int rdy = __builtin_amdgcn_readfirstlane(st[0]);
            const u32 hold = __builtin_amdgcn_readfirstlane(st[1]);
            if (rdy > n && !hold) break;
            __builtin_amdgcn_s_sleep(2);
            peek();
        }
        asm volatile("" ::: "memory");
    };
    auto load_step = [&](int n, bf16x8 (&fx)[4], bf16x8 (&fw)[4]) {
#pragma unroll
        for (int i = 0; i < 4; ++i) fx[i] = __builtin_bit_cast(bf16x8, ld16<GEMM_AUX>(rsm, loff + i * 1024, xa + n * STEP_STRIDE));
#pragma unroll
        for (int i = 0; i < 4; ++i) fw[i] = __builtin_bit_cast(bf16x8, ld16<GEMM_AUX>(rsm, loff + i * 1024, wa + n * STEP_STRIDE));
    };
    auto mma_step = [&](bf16x8 (&fx)[4], bf16x8 (&fw)[4]) {
#pragma unroll
        for (int a = 0; a < 4; ++a)
#pragma unroll
            for (int b = 0; b < 4; ++b)
                acc[a][b] = __builtin_amdgcn_mfma_f32_16x16x32_bf16(fw[b], fx[a], acc[a][b], 0, 0, 0);
    };
#pragma unroll
    for (int s = 0; s < 3; ++s) { wait_step(s); load_step(s, fxr[s], fwr[s]); }
    STAMP(1);
    for (int n = 0; n < 60; n += 4) {
#pragma unroll
        for (int s = 0; s < 4; ++s) {
            wait_step(n + s + 3);
            load_step(n + s + 3, fxr[(s + 3) & 3], fwr[(s + 3) & 3]);
            peek();
            __builtin_amdgcn_sched_barrier(0);
            mma_step(fxr[s], fwr[s]);
        }
    }
    wait_step(63);
    if (wave == 0 && lane == 0) lds_st(&sh.near, 1);
    load_step(63, fxr[3], fwr[3]);
    __builtin_amdgcn_sched_barrier(0);
#pragma unroll
    for (int s = 0; s < 4; ++s) mma_step(fxr[s], fwr[s]);
    STAMP(2);

    {
        u32x4 pk[8];
#pragma unroll
        for (int mt = 0; mt < 4; ++mt)
#pragma unroll
            for (int h = 0; h < 2; ++h) {
                pk[mt * 2 + h][0] = pack_bf16(acc[mt][0][2 * h], acc[mt][1][2 * h]);
                pk[mt * 2 + h][1] = pack_bf16(acc[mt][2][2 * h], acc[mt][3][2 * h]);
                pk[mt * 2 + h][2] = pack_bf16(acc[mt][0][2 * h + 1], acc[mt][1][2 * h + 1]);
                pk[mt * 2 + h][3] = pack_bf16(acc[mt][2][2 * h + 1], acc[mt][3][2 * h + 1]);
            }
        while (!lds_ld(&sh.gate)) __builtin_amdgcn_s_sleep(2);
        wait_until(lds_ld(&sh.slot) + ((wave * DELTA_Q4) >> 5));
        STAMP(3);
        char* const dws = p.ws[peer];
        const rsrc_t rsd = mk_rsrc(dws, FLAG_OFF);
        const u32 d = par + (u32)CB_OFF + ((rank * 32 + tau) * 4 + wave) * 8192;
#pragma unroll
        for (int i = 0; i < 8; ++i) st16_sys(rsd, loff + (i & 3) * 1024, d + (i >> 2) * 4096, pk[i]);
        WAIT_VM(0);
        if (lane == 0) {
            const u32 t = __hip_atomic_fetch_add(&sh.cdone[wr], 1u, __ATOMIC_RELAXED, __HIP_MEMORY_SCOPE_WORKGROUP);
            if (t == 1) st_sys(dws + PRIV_OFF + (size_t)(tau * 8 + (rank >> 1) * 2 + wr) * FROW + 8 + (rank & 1) * 4, call);
        }
        STAMP(4);
    }

    {
        while (!lds_ld(&sh.cready)) __builtin_amdgcn_s_sleep(2);
        asm volatile("" ::: "memory");
        STAMP(5);
        const bool bad = lds_ld(&sh.abort_) != 0;
        const int rwr = grp & 1, mt = wave;
        const int orow = (quad >> 1) * 256 + rb * 128 + rwr * 64 + mt * 16 + (lane & 15);
        const int ocol = (quad & 1) * 2048 + cb * 128 + (lane >> 4) * 32;
        const rsrc_t rso = mk_rsrc(p.out, 4 * MIB);
        const u32 o = orow * 8192 + ocol * 2;
        if (bad) {
            const u32x4 nanv = {0x7fc07fc0u, 0x7fc07fc0u, 0x7fc07fc0u, 0x7fc07fc0u};
#pragma unroll
            for (int r = 0; r < 4; ++r) __builtin_amdgcn_raw_buffer_store_b128(nanv, rso, o + r * 16, 0, 0);
            return;
        }
        u32x4 v[2][2][2];
#pragma unroll
        for (int s = 0; s < 2; ++s)
#pragma unroll
            for (int c = 0; c < 2; ++c)
#pragma unroll
                for (int h = 0; h < 2; ++h)
                    v[s][c][h] = ld16<GEMM_AUX>(rsm, loff + h * 1024, par + (u32)CB_OFF + (((quad * 2 + s) * 32 + tau) * 4 + rwr * 2 + c) * 8192 + mt * 2048);
#pragma unroll
        for (int r = 0; r < 4; ++r) {
            const int h = r >> 1, e = (r & 1) * 2;
            u32x4 res;
#pragma unroll
            for (int c = 0; c < 2; ++c)
#pragma unroll
                for (int z = 0; z < 2; ++z) {
                    const u32 a = v[0][c][h][e + z], b = v[1][c][h][e + z];
                    res[c * 2 + z] = pack_bf16(gelu_f(bf_lo(a) + bf_lo(b)), gelu_f(bf_hi(a) + bf_hi(b)));
                }
            __builtin_amdgcn_raw_buffer_store_b128(res, rso, o + r * 16, 0, OUT_AUX);
        }
        STAMP(6);
    }
}

extern "C" void kernel_launch(int rank, int nranks, ncclComm_t comm,
                              hipStream_t stream,
                              void* const* d_in, const int* in_sizes, int n_in,
                              void* d_out, int out_size,
                              const int* in_dtypes, int out_dtype,
                              void* const* d_out_all,
                              void* const* d_ws_all, size_t ws_bytes) {
    (void)comm; (void)in_dtypes; (void)out_dtype; (void)d_out_all;
    if (nranks != NR || (unsigned)rank >= (unsigned)NR || n_in != 2 || ws_bytes < WS_NEED ||
        in_sizes[0] != 4096 * 512 || in_sizes[1] != 512 * 4096 || out_size != 512 * 4096) {
        fprintf(stderr, "kernel_launch: unexpected problem shape\n");
        return;
    }
    Params p;
    p.x = (const char*)d_in[0];
    p.w = (const char*)d_in[1];
    p.out = (char*)d_out;
    for (int i = 0; i < NR; ++i) p.ws[i] = (char*)d_ws_all[i];
    p.rank = rank;
    int n = 0;
    for (int ks = 0; ks < 8; ++ks) {
        for (int i = 1; i <= 8; ++i) p.order[n++] = (u32)(ks * 8 + (i & 7));
        const int ng = ks < 8 - DEFER ? 8 : 4;
        for (int g = 0; g < ng; ++g) p.order[n++] = (u32)(64 + ks * 8 + g);
    }
    for (int ks = 8 - DEFER; ks < 8; ++ks)
        for (int g = 4; g < 8; ++g) p.order[n++] = (u32)(64 + ks * 8 + g);
    hipLaunchKernelGGL(fused_gemm_rs, dim3(NWG), dim3(NTHREADS), 0, stream, p);
    HIP_CHECK(hipGetLastError());
}
```

```cpp
#include <hip/hip_runtime.h>
#include <rccl/rccl.h>
#include <cstdio>
#include <cstdint>

#define HIP_CHECK(expr) do { hipError_t _e = (expr); if (_e != hipSuccess) { \
    fprintf(stderr, "HIP error %s at %s:%d\n", hipGetErrorString(_e), __FILE__, __LINE__); \
    return; } } while (0)

typedef unsigned int u32;
typedef unsigned long long u64;
typedef u32 u32x4 __attribute__((ext_vector_type(4)));
typedef u32 u32x2 __attribute__((ext_vector_type(2)));
typedef float f32x4 __attribute__((ext_vector_type(4)));
typedef __bf16 bf16x8 __attribute__((ext_vector_type(8)));
typedef __bf16 bf16x2 __attribute__((ext_vector_type(2)));
typedef float f32x2 __attribute__((ext_vector_type(2)));
#define GAS __attribute__((address_space(1)))

constexpr int NR = 8;
constexpr int NWG = 256;
constexpr int NTHREADS = 384;
#define PUSHERS 32
#define DEFER {0, 0, 2, 3, 4, 4, 5, 6}
#define ANN 60
#define DELTA_Q4 448
#define DELTA_C 444
#define TOUCH 0
#define HOLD_MODE 1
#define ACK_OK 220
#define SHIFT_CAP 30
#define SHIFT_LEAD 110
#define FAST_SLOT 28
#define PIPE_POLL 4
#define EAGER 6
#define LAZY 40
#define DIAG_DELAY 60
#define DIAG_GAP 20
#define START_TICKS 90
#define HOLD_PRE 30
#define NEAR_TICKS 100
#define FROW 4352
#define POLL_SLEEP 8
#define MASTER_SLEEP 4
#define CPOLL_SLEEP 8
#define GEMM_AUX 0
#define OUT_AUX 17
#define LAYOUT 1
#define IDX_STRIDE 16384
#define STEP_STRIDE (8 * 16384)
constexpr int NUNITS = 120;
constexpr size_t MIB = 1u << 20;
constexpr size_t XB_OFF = 0, WB_OFF = 8 * MIB, CB_OFF = 16 * MIB, PAR_BYTES = 24 * MIB;
constexpr size_t FLAG_OFF = 2 * PAR_BYTES;
constexpr size_t FX_OFF = FLAG_OFF;
constexpr size_t FW_OFF = FLAG_OFF + 8 * FROW;
constexpr size_t GATE_OFF = FLAG_OFF + 16 * FROW;
constexpr size_t PRIV_OFF = GATE_OFF + 8 * FROW;
constexpr size_t SEQ_OFF = PRIV_OFF + NWG * FROW;
constexpr size_t DBG_OFF = SEQ_OFF + NWG * 8 * 4;
constexpr size_t WS_NEED = DBG_OFF + NWG * 8 * 8 * 8;
constexpr u64 DEADLINE_TICKS = 40000000ull;

struct Params {
    const char* x;
    const char* w;
    char* out;
    int rank, pad;
    char* ws[NR];
    u32 order[NUNITS];
};

typedef __amdgpu_buffer_rsrc_t rsrc_t;
__device__ __forceinline__ rsrc_t mk_rsrc(const void* p, size_t bytes) {
    return __builtin_amdgcn_make_buffer_rsrc((void*)p, (short)0, (int)bytes, 0x00020000);
}
constexpr int AUX_SYS = 17;
__device__ __forceinline__ void st16_sys(rsrc_t r, u32 voff, u32 soff, u32x4 v) {
    __builtin_amdgcn_raw_buffer_store_b128(v, r, voff, soff, AUX_SYS);
}
template <int AUX = 0>
__device__ __forceinline__ u32x4 ld16(rsrc_t r, u32 voff, u32 soff) {
    return __builtin_amdgcn_raw_buffer_load_b128(r, voff, soff, AUX);
}
__device__ __forceinline__ u32 ld_sys(const char* p) {
    return __hip_atomic_load((const GAS u32*)p, __ATOMIC_RELAXED, __HIP_MEMORY_SCOPE_SYSTEM);
}
__device__ __forceinline__ void st_sys(char* p, u32 v) {
    __hip_atomic_store((GAS u32*)p, v, __ATOMIC_RELAXED, __HIP_MEMORY_SCOPE_SYSTEM);
}
__device__ __forceinline__ u32 ld_agent(const char* p) {
    return __hip_atomic_load((const GAS u32*)p, __ATOMIC_RELAXED, __HIP_MEMORY_SCOPE_AGENT);
}
__device__ __forceinline__ void st_agent(char* p, u32 v) {
    __hip_atomic_store((GAS u32*)p, v, __ATOMIC_RELAXED, __HIP_MEMORY_SCOPE_AGENT);
}
#define WAIT_VM(n) asm volatile("s_waitcnt vmcnt(" #n ")" ::: "memory")

#define STAMP(i) do {} while (0)
#define STAMPD(i, d) do {} while (0)

struct __attribute__((aligned(16))) Shared {
    u32 ready;
    u32 hold;
    u32 gate;
    u32 slot;
    u32 near;
    u32 cready;
    u32 abort_;
    u32 sync;
    u32x4 stage[16][8][64];
};

#define LAS __attribute__((address_space(3)))
__device__ __forceinline__ u32 lds_ld(u32* p) { return __builtin_amdgcn_readfirstlane(*(volatile LAS u32*)p); }
__device__ __forceinline__ void lds_st(u32* p, u32 v) { *(volatile LAS u32*)p = v; }

__device__ __forceinline__ void wait_until(u32 t) {
    for (;;) {
        const int rem = (int)(t - (u32)__builtin_amdgcn_s_memrealtime());
        if (rem <= 0) break;
        __builtin_amdgcn_s_sleep(1);
    }
}

__device__ __forceinline__ u32 link_shift(const char* ws, int g, u32 call) {
    const u64 v = __hip_atomic_load((const GAS u64*)(ws + GATE_OFF + g * FROW + 128), __ATOMIC_RELAXED, __HIP_MEMORY_SCOPE_AGENT);
    return __builtin_amdgcn_readfirstlane((u32)(v >> 16) == call ? (u32)v & 0xffffu : 0u);
}

__device__ __forceinline__ u32 pack_bf16(float a, float b) {
    f32x2 v = {a, b};
    bf16x2 r = __builtin_convertvector(v, bf16x2);
    return __builtin_bit_cast(u32, r);
}
__device__ __forceinline__ f32x2 gelu2(f32x2 y) {
    const float c2 = -2.0f * 0.7978845608028654f * 1.4426950408889634f;
    const f32x2 k1 = {c2 * 0.044715f, c2 * 0.044715f}, k0 = {c2, c2}, one = {1.0f, 1.0f};
    const f32x2 t = y * __builtin_elementwise_fma(y * y, k1, k0);
    const f32x2 den = one + f32x2{__builtin_amdgcn_exp2f(t[0]), __builtin_amdgcn_exp2f(t[1])};
    return y * f32x2{__builtin_amdgcn_rcpf(den[0]), __builtin_amdgcn_rcpf(den[1])};
}
__device__ __forceinline__ float bf_lo(u32 v) { return __builtin_bit_cast(float, v << 16); }
__device__ __forceinline__ float bf_hi(u32 v) { return __builtin_bit_cast(float, v & 0xffff0000u); }

__device__ __forceinline__ void transpose8x8(u32x4 (&v)[8]) {
    u32x4 o[8];
#pragma unroll
    for (int c = 0; c < 4; ++c) {
#pragma unroll
        for (int r = 0; r < 4; ++r) {
            u32 a = v[2 * r][c], b = v[2 * r + 1][c];
            o[2 * c][r] = __builtin_amdgcn_perm(b, a, 0x05040100u);
            o[2 * c + 1][r] = __builtin_amdgcn_perm(b, a, 0x07060302u);
        }
    }
#pragma unroll
    for (int i = 0; i < 8; ++i) v[i] = o[i];
}

__global__ void __launch_bounds__(NTHREADS, 2) fused_gemm_rs(Params p) {
    __shared__ Shared sh;
    const int lane = threadIdx.x & 63;
    const int wave = __builtin_amdgcn_readfirstlane(threadIdx.x >> 6);
    const int grp = blockIdx.x & 7;
    const int tau = blockIdx.x >> 3;
    const int rank = p.rank;
    const int peer = (rank ^ grp) & 7;
    char* const myws = p.ws[0];
    char* const dws = p.ws[grp];
    const char* const px = p.x;
    const char* const pw_ = p.w;
    const int order0 = p.order[tau];
    asm volatile("" :: "s"(rank), "s"(myws), "s"(dws), "s"(px), "s"(pw_), "s"(order0));
    const u32 tstart = (u32)__builtin_amdgcn_s_memrealtime() + START_TICKS;

    if (wave >= 4) __builtin_amdgcn_s_setprio(3);
    char* const seqp = myws + SEQ_OFF + (size_t)(blockIdx.x * 8 + wave) * 4;
    if (threadIdx.x == 0) { sh.ready = 0; sh.hold = 0; sh.gate = 0; sh.near = 0; sh.cready = 0; sh.abort_ = 0; sh.sync = 0; }
    asm volatile("s_waitcnt lgkmcnt(0)" ::: "memory");

    if (wave == 4) {
        const int pw = tau;
        const int pi = (peer >> 2) & 1, pj = (peer >> 1) & 1, pk = peer & 1;
        const int q = lane >> 4, r = lane & 15;
        const rsrc_t rsx = mk_rsrc(px, 4 * MIB), rsw = mk_rsrc(pw_, 4 * MIB);
        const u32 xv = (pi * 256 + r) * 1024 + pk * 512 + q * 16;
        const u32 wv = (pk * 256 + q * 8) * 8192 + pj * 4096 + r * 16;
        u32x4 va[8], vb[8];
        auto issue_loads = [&](int o) {
            const int ks = (o >> 3) & 7, idx = o & 7;
            if (o & 64) {
                const u32 so = ks * 32 * 8192 + idx * 512;
#pragma unroll
                for (int i = 0; i < 8; ++i) { va[i] = ld16(rsw, wv, so + i * 8192); vb[i] = ld16(rsw, wv + 256, so + i * 8192); }
            } else {
                const u32 so = ((rank ^ idx) & 7) * 512 * 1024 + ks * 64;
#pragma unroll
                for (int i = 0; i < 8; ++i) { va[i] = ld16(rsx, xv, so + i * 16384); vb[i] = ld16(rsx, xv, so + (i + 8) * 16384); }
            }
        };
        u32 touched = 0;
        const bool eager = grp != 0 && pw < EAGER;
        const u32 seq_raw = ld_agent(seqp);
        if (eager) issue_loads(order0);
        __builtin_amdgcn_s_barrier();
        if (!eager) {
            wait_until(tstart - LAZY + (grp != 0 ? ((pw - EAGER) * DELTA_Q4) >> 5 : 0));
            issue_loads(order0);
        }
        const u32 call = __builtin_amdgcn_readfirstlane(seq_raw) + 1;
        if (lane == 0) st_agent(seqp, call);
        const rsrc_t rsd = mk_rsrc(dws, FLAG_OFF);
        const u32 dpar = (call & 1) * (u32)PAR_BYTES;
        auto clear = [&](int k) {
            const rsrc_t rsm = mk_rsrc(myws, FLAG_OFF);
            const u32x4 zero = {0, 0, 0, 0};
#pragma unroll
            for (int i = 0; i < 8; ++i)
                st16_sys(rsm, lane * 16 + (i & 3) * 1024, (u32)PAR_BYTES - dpar + (u32)CB_OFF + (blockIdx.x * 4 + k) * 8192 + (i >> 2) * 4096, zero);
        };
        u32 shift = 0;
        for (int u = pw; u < NUNITS; u += PUSHERS) {
            const int o = p.order[u];
            const bool isw = o & 64;
            const int ks = (o >> 3) & 7, idx = isw ? (o & 7) : ((o ^ grp) & 7);
            const int n = grp == 0 ? ks : 8 + ks * 7 + (grp - 1);
            if (isw) { transpose8x8(va); transpose8x8(vb); }
            const u32 d = dpar + (u32)(isw ? WB_OFF : XB_OFF) + idx * IDX_STRIDE + n * STEP_STRIDE;
            const u32 nominal = tstart + ((u * DELTA_Q4) >> 4);
            if (grp != 0) {
                if (u >= 8) {
                    for (int i = 0; i < 8; ++i) {
                        wait_until(nominal + shift - SHIFT_LEAD);
                        const u32 old = shift;
                        shift = link_shift(myws, grp, call);
                        if (shift < old + 40) break;
                    }
                }
                wait_until(nominal + shift - HOLD_PRE);
                if (HOLD_MODE < 2 && lane == 0) lds_st(&sh.hold, 1);
                wait_until(nominal + shift);
            }
#pragma unroll
            for (int i = 0; i < 8; ++i) st16_sys(rsd, lane * 16 + (i & 3) * 1024, d + (i >> 2) * 4096, va[i]);
#pragma unroll
            for (int i = 0; i < 8; ++i) st16_sys(rsd, lane * 16 + (i & 3) * 1024, d + 8192 + (i >> 2) * 4096, vb[i]);
            if (HOLD_MODE == 1 && lane == 0) lds_st(&sh.hold, 0);
            if (u + PUSHERS < NUNITS) {
                issue_loads(p.order[u + PUSHERS]);
                WAIT_VM(16);
            } else {
                WAIT_VM(0);
            }
            if (grp != 0) {
                const int over = (int)((u32)__builtin_amdgcn_s_memrealtime() - ACK_OK - nominal - shift);
                if (over > 0 && lane == 0)
                    __hip_atomic_fetch_max((GAS u64*)(myws + GATE_OFF + grp * FROW + 128), ((u64)call << 16) + min(shift + min(over, SHIFT_CAP), 4000u),
                                           __ATOMIC_RELAXED, __HIP_MEMORY_SCOPE_AGENT);
            }
            if (lane == 0) { st_sys(dws + (isw ? FW_OFF : FX_OFF) + idx * FROW + n * 4, call); lds_st(&sh.hold, 0); }
            clear(u / PUSHERS);
        }
        if (pw + 3 * PUSHERS >= NUNITS) clear(3);
        asm volatile("" :: "v"(touched));
        STAMPD(7, shift);
        STAMP(2);
        return;
    }

    __builtin_amdgcn_s_barrier();
    const u32 call = __builtin_amdgcn_readfirstlane(ld_agent(seqp)) + 1;
    if (lane == 0) st_agent(seqp, call);
    const int rb = tau & 1, cb = tau >> 1;
    const int quad = grp >> 1;

    if (wave == 5) {
        const u64 t0 = __builtin_amdgcn_s_memrealtime();
        bool dead = false;
        u32 polls = 0;
        const char* priv = myws + PRIV_OFF + (size_t)blockIdx.x * FROW + (lane & 7) * 4;
        auto expired = [&](u32 pv) -> bool {
            if (__builtin_amdgcn_readlane(pv, 4) == call) return true;
            if ((++polls & 255) == 0 && __builtin_amdgcn_s_memrealtime() - t0 > DEADLINE_TICKS) {
                for (int i = 0; i < 4; ++i) st_agent(myws + PRIV_OFF + (size_t)(i * 64 + lane) * FROW + 16, call);
                return true;
            }
            return false;
        };
        bool master = grp == 0 && tau < 16;
        const char* frow = myws + FX_OFF + tau * FROW + lane * 4;
        const int cons = tau < 8 ? (lane & 31) * 8 + tau : (tau - 8) * 32 + (lane & 31);
        char* const pub_w = myws + PRIV_OFF + (size_t)cons * FROW + (tau >> 3) * 4;
        const bool skip = tau < 8 && (lane < 8 ? tau == 0 : (lane - 8) % 7 == tau - 1);
        u32 pubd = 0, seen = 0;
        for (;;) {
            u32 a = 0;
            if (master) a = ld_sys(frow);
            const u32 pv = ld_sys(priv);
            if (master) {
                const u64 m = ~__ballot(a >= call || skip);
                const u32 c = m ? (u32)__builtin_ctzll(m) : 64u;
                if (c != pubd) {
                    pubd = c;
                    if (lane < 32) st_agent(pub_w, call * 128 + c);
                    if (c == 64) master = false;
                }
            }
            const int c0 = (int)(__builtin_amdgcn_readlane(pv, 0) - call * 128), c1 = (int)(__builtin_amdgcn_readlane(pv, 1) - call * 128);
            const int c = min(c0, c1);
            if (c > (int)seen) {
                seen = c;
                if (lane == 0) lds_st(&sh.ready, seen);
            }
            if (seen == 64 && !master) break;
            if (expired(pv)) { dead = true; break; }
            if (master) __builtin_amdgcn_s_sleep(MASTER_SLEEP); else __builtin_amdgcn_s_sleep(POLL_SLEEP);
        }
        STAMP(1);
        if (dead && lane == 0) { lds_st(&sh.abort_, 1); lds_st(&sh.ready, 64); }
        while (!lds_ld(&sh.near)) __builtin_amdgcn_s_sleep(2);
        wait_until(tstart + ((NUNITS * DELTA_Q4) >> 4) - 250);
        const u32 shift = grp != 0 ? link_shift(myws, grp, call) : 0;
        if (lane == 0) {
            const u64 now = __builtin_amdgcn_s_memrealtime();
            u64 slot = now;
            if (grp != 0) {
                u64 first = now + (u64)(long long)(int)(tstart + ((NUNITS * DELTA_Q4) >> 4) + shift - (u32)now);
                u64 lo = now + NEAR_TICKS;
                if ((long long)(first - lo) > 0) lo = first;
                GAS u64* clk = (GAS u64*)(myws + GATE_OFF + grp * FROW);
                __hip_atomic_fetch_max(clk, lo << 3, __ATOMIC_RELAXED, __HIP_MEMORY_SCOPE_AGENT);
                slot = __hip_atomic_fetch_add(clk, (u64)DELTA_C, __ATOMIC_RELAXED, __HIP_MEMORY_SCOPE_AGENT) >> 3;
                if (slot - now > 100000ull) slot = now;
            }
            lds_st(&sh.slot, (u32)slot);
            lds_st(&sh.gate, 1);
        }
        for (int h = 0; h < 2; ++h) {
            wait_until(lds_ld(&sh.slot) + (((2 * h + 1) * DELTA_C) >> 5) - ANN);
            if (lane == 0) st_sys(dws + PRIV_OFF + (size_t)(tau * 8 + (rank >> 1) * 2 + h) * FROW + 8 + (rank & 1) * 4, call);
        }
        auto done = [&](u32 pv) { return __builtin_amdgcn_readlane(pv, 2) >= call && __builtin_amdgcn_readlane(pv, 3) >= call; };
        const u32 tfast = tstart + ((NUNITS * DELTA_Q4) >> 4) + ((FAST_SLOT * DELTA_C) >> 3);
        bool got = dead;
        while (!got && (int)((u32)__builtin_amdgcn_s_memrealtime() - tfast) < 0) {
            got = done(ld_sys(priv));
            __builtin_amdgcn_s_sleep(CPOLL_SLEEP);
        }
        if (!got) {
            u32 p0 = ld_sys(priv);
            __builtin_amdgcn_s_sleep(PIPE_POLL);
            u32 p1 = ld_sys(priv);
            __builtin_amdgcn_s_sleep(PIPE_POLL);
            u32 p2 = ld_sys(priv), p3;
            for (;;) {
                __builtin_amdgcn_s_sleep(PIPE_POLL);
                p3 = ld_sys(priv);
                __builtin_amdgcn_sched_barrier(0);
                if (done(p0)) break;
                __builtin_amdgcn_s_sleep(PIPE_POLL);
                p0 = ld_sys(priv);
                __builtin_amdgcn_sched_barrier(0);
                if (done(p1)) break;
                __builtin_amdgcn_s_sleep(PIPE_POLL);
                p1 = ld_sys(priv);
                __builtin_amdgcn_sched_barrier(0);
                if (done(p2)) break;
                __builtin_amdgcn_s_sleep(PIPE_POLL);
                p2 = ld_sys(priv);
                __builtin_amdgcn_sched_barrier(0);
                if (done(p3)) break;
                if (expired(p3)) { dead = true; break; }
            }
        }
        if (dead && lane == 0) lds_st(&sh.abort_, 1);
        if (lane == 0) lds_st(&sh.cready, 1);
        STAMP(3);
        STAMPD(0, -40);
        return;
    }

    const int wr = wave >> 1, wc = wave & 1;
    const rsrc_t rsm = mk_rsrc(myws, FLAG_OFF);
    const u32 par = (call & 1) * (u32)PAR_BYTES;
    const u32 xa = par + (u32)XB_OFF + grp * IDX_STRIDE + (rb * 8 + wr * 4) * 1024;
    const u32 wa = par + (u32)WB_OFF + (cb >> 1) * IDX_STRIDE + ((cb & 1) * 8 + wc * 4) * 1024;
    const u32 loff = lane * 16;
    u32x2 st = {0, 0};
    auto peek = [&]() { st = *(volatile LAS u32x2*)&sh.ready; };
    auto wait_step = [&](int n) {
        for (;;) {
            const int rdy = __builtin_amdgcn_readfirstlane(st[0]);
            const u32 hold = __builtin_amdgcn_readfirstlane(st[1]);
            if (rdy > n && !hold) break;
            __builtin_amdgcn_s_sleep(2);
            peek();
        }
        asm volatile("" ::: "memory");
    };
    const int orow = (quad >> 1) * 256 + rb * 128 + (grp & 1) * 64 + wave * 16 + (lane & 15);
    const u32 cva = (u32)CB_OFF + ((quad * 2 * 32 + tau) * 4 + (grp & 1) * 2) * 8192 + wave * 2048;
    {
        wait_until(tstart + DIAG_DELAY);
        const rsrc_t rsx = mk_rsrc(px, 4 * MIB), rsw = mk_rsrc(pw_, 4 * MIB);
        const u32 xv = (rank * 512 + orow) * 1024 + (lane >> 4) * 16;
        const u32 wv = (lane >> 4) * 8 * 8192 + (quad & 1) * 4096 + cb * 256 + (lane & 15) * 16;
        auto meet = [&](u32 target) {
            asm volatile("s_waitcnt lgkmcnt(0)" ::: "memory");
            if (lane == 0) __hip_atomic_fetch_add(&sh.sync, 1u, __ATOMIC_RELAXED, __HIP_MEMORY_SCOPE_WORKGROUP);
            while (lds_ld(&sh.sync) < target) __builtin_amdgcn_s_sleep(1);
            asm volatile("" ::: "memory");
        };
        u32x4 wb[4][8], xb[16];
#pragma unroll
        for (int t = 0; t < 4; ++t) {
            peek();
            wait_step(-1);
#pragma unroll
            for (int i = 0; i < 8; ++i) wb[t][i] = ld16(rsw, wv, ((wave * 4 + t) * 32 + i) * 8192);
#pragma unroll
            for (int i = 0; i < 4; ++i) xb[t * 4 + i] = ld16(rsx, xv + (t * 4 + i) * 64, 0);
            if (DIAG_GAP) __builtin_amdgcn_s_sleep(DIAG_GAP);
        }
#pragma unroll
        for (int t = 0; t < 4; ++t) {
            transpose8x8(wb[t]);
#pragma unroll
            for (int i = 0; i < 8; ++i) sh.stage[wave * 4 + t][i][lane] = wb[t][i];
        }
        meet(4);
        f32x4 d[8];
#pragma unroll
        for (int i = 0; i < 8; ++i) d[i] = f32x4{0.f, 0.f, 0.f, 0.f};
#pragma unroll
        for (int k = 0; k < 17; ++k) {
            if (k < 16) {
#pragma unroll
                for (int i = 0; i < 8; ++i) wb[k & 1][i] = sh.stage[k][i][lane];
            }
            __builtin_amdgcn_sched_barrier(0);
            if (k > 0) {
#pragma unroll
                for (int i = 0; i < 8; ++i)
                    d[i] = __builtin_amdgcn_mfma_f32_16x16x32_bf16(__builtin_bit_cast(bf16x8, wb[(k - 1) & 1][i]), __builtin_bit_cast(bf16x8, xb[k - 1]), d[i], 0, 0, 0);
            }
            __builtin_amdgcn_sched_barrier(0);
        }
        meet(8);
#pragma unroll
        for (int i = 0; i < 8; ++i) {
            const int r = i >> 1, c = (i & 1) * 4;
            sh.stage[wave][i][lane] = __builtin_bit_cast(u32x4, f32x4{d[c][r], d[c + 1][r], d[c + 2][r], d[c + 3][r]});
        }
        STAMP(4);
    }
    f32x4 acc[4][4];
#pragma unroll
    for (int a = 0; a < 4; ++a)
#pragma unroll
        for (int b = 0; b < 4; ++b) acc[a][b] = f32x4{0.f, 0.f, 0.f, 0.f};
    bf16x8 fxr[4][4], fwr[4][4];
    int n = grp == 0 ? 8 : 0, nskip = grp == 0 ? 99 : 7 + grp;
    auto load_step = [&](bf16x8 (&fx)[4], bf16x8 (&fw)[4]) {
        wait_step(n);
#pragma unroll
        for (int i = 0; i < 4; ++i) fx[i] = __builtin_bit_cast(bf16x8, ld16<GEMM_AUX>(rsm, loff + i * 1024, xa + n * STEP_STRIDE));
#pragma unroll
        for (int i = 0; i < 4; ++i) fw[i] = __builtin_bit_cast(bf16x8, ld16<GEMM_AUX>(rsm, loff + i * 1024, wa + n * STEP_STRIDE));
        ++n;
        if (n == nskip) { ++n; nskip += 7; }
    };
    auto mma_step = [&](bf16x8 (&fx)[4], bf16x8 (&fw)[4]) {
#pragma unroll
        for (int a = 0; a < 4; ++a)
#pragma unroll
            for (int b = 0; b < 4; ++b)
                acc[a][b] = __builtin_amdgcn_mfma_f32_16x16x32_bf16(fw[b], fx[a], acc[a][b], 0, 0, 0);
    };
#pragma unroll
    for (int s = 0; s < 3; ++s) load_step(fxr[s], fwr[s]);
    for (int m = 0; m < 52; m += 4) {
#pragma unroll
        for (int s = 0; s < 4; ++s) {
            load_step(fxr[(s + 3) & 3], fwr[(s + 3) & 3]);
            peek();
            __builtin_amdgcn_sched_barrier(0);
            mma_step(fxr[s], fwr[s]);
        }
    }
    load_step(fxr[3], fwr[3]);
    if (wave == 0 && lane == 0) lds_st(&sh.near, 1);
    __builtin_amdgcn_sched_barrier(0);
#pragma unroll
    for (int s = 0; s < 4; ++s) mma_step(fxr[s], fwr[s]);
    STAMP(2);

    {
        u32x4 pk[8];
#pragma unroll
        for (int mt = 0; mt < 4; ++mt)
#pragma unroll
            for (int h = 0; h < 2; ++h) {
                pk[mt * 2 + h][0] = pack_bf16(acc[mt][0][2 * h], acc[mt][1][2 * h]);
                pk[mt * 2 + h][1] = pack_bf16(acc[mt][2][2 * h], acc[mt][3][2 * h]);
                pk[mt * 2 + h][2] = pack_bf16(acc[mt][0][2 * h + 1], acc[mt][1][2 * h + 1]);
                pk[mt * 2 + h][3] = pack_bf16(acc[mt][2][2 * h + 1], acc[mt][3][2 * h + 1]);
            }
#pragma unroll
        for (int i = 0; i < 8; ++i)
#pragma unroll
            for (int e = 0; e < 4; ++e) pk[i][e] = pk[i][e] ? pk[i][e] : 0x8000u;
        while (!lds_ld(&sh.gate)) __builtin_amdgcn_s_sleep(2);
        wait_until(lds_ld(&sh.slot) + ((wave * DELTA_C) >> 5));
        STAMP(3);
        const rsrc_t rsd = mk_rsrc(dws, FLAG_OFF);
        const u32 dst = par + (u32)CB_OFF + ((rank * 32 + tau) * 4 + wave) * 8192;
#pragma unroll
        for (int i = 0; i < 8; ++i) st16_sys(rsd, loff + (i & 3) * 1024, dst + (i >> 2) * 4096, pk[i]);
    }

    {
        f32x4 d[8];
#pragma unroll
        for (int i = 0; i < 8; ++i) d[i] = __builtin_bit_cast(f32x4, sh.stage[wave][i][lane]);
#pragma unroll
        for (int i = 0; i < 8; ++i) asm volatile("" : "+v"(d[i]));
        while (!lds_ld(&sh.cready)) __builtin_amdgcn_s_sleep(1);
        asm volatile("" ::: "memory");
        bool bad = lds_ld(&sh.abort_) != 0;
        u32x4 v[2][2][2];
        while (!bad) {
            u32 mn = ~0u;
#pragma unroll
            for (int s = 0; s < 2; ++s)
#pragma unroll
                for (int c = 0; c < 2; ++c)
#pragma unroll
                    for (int h = 0; h < 2; ++h)
                        v[s][c][h] = ld16<AUX_SYS>(rsm, loff + h * 1024, par + cva + (s * 32 * 4 + c) * 8192);
#pragma unroll
            for (int i = 0; i < 8; ++i)
#pragma unroll
                for (int e = 0; e < 4; ++e) mn = min(mn, v[i >> 2][(i >> 1) & 1][i & 1][e]);
            if (__ballot(mn == 0) == 0) break;
            if ((u32)__builtin_amdgcn_s_memrealtime() - tstart > 2 * (u32)DEADLINE_TICKS) bad = true;
        }
        const int ocol = (quad & 1) * 2048 + cb * 128 + (lane >> 4) * 32;
        const rsrc_t rso = mk_rsrc(p.out, 4 * MIB);
        const u32 o = orow * 8192 + ocol * 2;
        if (bad) {
            const u32x4 nanv = {0x7fc07fc0u, 0x7fc07fc0u, 0x7fc07fc0u, 0x7fc07fc0u};
#pragma unroll
            for (int r = 0; r < 4; ++r) __builtin_amdgcn_raw_buffer_store_b128(nanv, rso, o + r * 16, 0, 0);
            return;
        }
#pragma unroll
        for (int r = 0; r < 4; ++r) {
            const int h = r >> 1, e = (r & 1) * 2;
            u32x4 res;
#pragma unroll
            for (int c = 0; c < 2; ++c)
#pragma unroll
                for (int z = 0; z < 2; ++z) {
                    const u32 a = v[0][c][h][e + z], b = v[1][c][h][e + z];
                    const f32x2 y = (f32x2{bf_lo(a), bf_hi(a)} + f32x2{bf_lo(b), bf_hi(b)}) + f32x2{d[r * 2 + c][z * 2], d[r * 2 + c][z * 2 + 1]};
                    res[c * 2 + z] = __builtin_bit_cast(u32, __builtin_convertvector(gelu2(y), bf16x2));
                }
            __builtin_amdgcn_raw_buffer_store_b128(res, rso, o + r * 16, 0, OUT_AUX);
        }
        STAMP(6);
        STAMPD(0, -40);
    }
}

extern "C" void kernel_launch(int rank, int nranks, ncclComm_t comm,
                              hipStream_t stream,
                              void* const* d_in, const int* in_sizes, int n_in,
                              void* d_out, int out_size,
                              const int* in_dtypes, int out_dtype,
                              void* const* d_out_all,
                              void* const* d_ws_all, size_t ws_bytes) {
    (void)comm; (void)in_dtypes; (void)out_dtype; (void)d_out_all;
    if (nranks != NR || (unsigned)rank >= (unsigned)NR || n_in != 2 || ws_bytes < WS_NEED ||
        in_sizes[0] != 4096 * 512 || in_sizes[1] != 512 * 4096 || out_size != 512 * 4096) {
        fprintf(stderr, "kernel_launch: unexpected problem shape\n");
        return;
    }
    Params p;
    p.x = (const char*)d_in[0];
    p.w = (const char*)d_in[1];
    p.out = (char*)d_out;
    for (int i = 0; i < NR; ++i) p.ws[i] = (char*)d_ws_all[rank ^ i];
    p.rank = rank;
    p.pad = 0;
    const int defer[8] = DEFER;
    int n = 0;
    for (int ks = 0; ks < 8; ++ks) {
        for (int i = 1; i < 8; ++i) p.order[n++] = (u32)(ks * 8 + i);
        for (int g = 0; g < 8; ++g)
            if (ks < 8 - defer[g]) p.order[n++] = (u32)(64 + ks * 8 + g);
    }
    for (int g = 0; g < 8; ++g)
        for (int ks = 8 - defer[g]; ks < 8; ++ks) p.order[n++] = (u32)(64 + ks * 8 + g);
    hipLaunchKernelGGL(fused_gemm_rs, dim3(NWG), dim3(NTHREADS), 0, stream, p);
    HIP_CHECK(hipGetLastError());
}
```

```cpp
#include <hip/hip_runtime.h>
#include <rccl/rccl.h>
#include <cstdio>
#include <cstdint>

#define HIP_CHECK(expr) do { hipError_t _e = (expr); if (_e != hipSuccess) { \
    fprintf(stderr, "HIP error %s at %s:%d\n", hipGetErrorString(_e), __FILE__, __LINE__); \
    return; } } while (0)

typedef unsigned int u32;
typedef unsigned long long u64;
typedef u32 u32x4 __attribute__((ext_vector_type(4)));
typedef u32 u32x2 __attribute__((ext_vector_type(2)));
typedef float f32x4 __attribute__((ext_vector_type(4)));
typedef __bf16 bf16x8 __attribute__((ext_vector_type(8)));
typedef __bf16 bf16x2 __attribute__((ext_vector_type(2)));
typedef float f32x2 __attribute__((ext_vector_type(2)));
#define GAS __attribute__((address_space(1)))

constexpr int NR = 8;
constexpr int NWG = 256;
constexpr int NTHREADS = 384;
#define PUSHERS 32
#define KEY_G 20
#define KEY_K 18
#define ANN 30
#define DELTA_Q4 450
#define DELTA_C 444
#define ACK_OK 170
#define SHIFT_CAP 300
#define SHIFT_LEAD 110
#define RSKEW 8
#define VOTES 4
#define DIAG_LATE 16
#define DRY 1
#define EAGER 6
#define LAZY 40
#define DIAG_GAP 8
#define START_TICKS 105
#define HOLD_PRE 30
#define NEAR_TICKS 100
#define FROW 4352
#define POLL_SLEEP 40
#define MASTER_SLEEP 4
#define CPOLL_SLEEP 2
#define OUT_AUX 17
#define STAG 16
#define IDX_STRIDE 16384
#define STEP_STRIDE (8 * 16384)
constexpr int NUNITS = 120;
constexpr size_t MIB = 1u << 20;
constexpr size_t XB_OFF = 0, WB_OFF = 8 * MIB, CB_OFF = 16 * MIB, PAR_BYTES = 24 * MIB;
constexpr size_t FLAG_OFF = 2 * PAR_BYTES;
constexpr size_t FX_OFF = FLAG_OFF;
constexpr size_t FW_OFF = FLAG_OFF + 8 * FROW;
constexpr size_t GATE_OFF = FLAG_OFF + 16 * FROW;
constexpr size_t PRIV_OFF = GATE_OFF + 8 * FROW;
constexpr size_t SEQ_OFF = PRIV_OFF + NWG * FROW;
constexpr size_t WS_NEED = SEQ_OFF + NWG * 8 * 4;
constexpr u64 DEADLINE_TICKS = 40000000ull;

struct Params {
    const char* x;
    const char* w;
    char* out;
    int rank, pad;
    char* ws[NR];
    u32 order[NUNITS];
};

typedef __amdgpu_buffer_rsrc_t rsrc_t;
__device__ __forceinline__ rsrc_t mk_rsrc(const void* p, size_t bytes) {
    return __builtin_amdgcn_make_buffer_rsrc((void*)p, (short)0, (int)bytes, 0x00020000);
}
constexpr int AUX_SYS = 17;
__device__ __forceinline__ void st16_sys(rsrc_t r, u32 voff, u32 soff, u32x4 v) {
    __builtin_amdgcn_raw_buffer_store_b128(v, r, voff, soff, AUX_SYS);
}
template <int AUX = 0>
__device__ __forceinline__ u32x4 ld16(rsrc_t r, u32 voff, u32 soff) {
    return __builtin_amdgcn_raw_buffer_load_b128(r, voff, soff, AUX);
}
__device__ __forceinline__ u32 ld_sys(const char* p) {
    return __hip_atomic_load((const GAS u32*)p, __ATOMIC_RELAXED, __HIP_MEMORY_SCOPE_SYSTEM);
}
__device__ __forceinline__ void st_sys(char* p, u32 v) {
    __hip_atomic_store((GAS u32*)p, v, __ATOMIC_RELAXED, __HIP_MEMORY_SCOPE_SYSTEM);
}
__device__ __forceinline__ u32 ld_agent(const char* p) {
    return __hip_atomic_load((const GAS u32*)p, __ATOMIC_RELAXED, __HIP_MEMORY_SCOPE_AGENT);
}
__device__ __forceinline__ void st_agent(char* p, u32 v) {
    __hip_atomic_store((GAS u32*)p, v, __ATOMIC_RELAXED, __HIP_MEMORY_SCOPE_AGENT);
}
#define WAIT_VM(n) asm volatile("s_waitcnt vmcnt(" #n ")" ::: "memory")


struct __attribute__((aligned(16))) Shared {
    u32 ready;
    u32 hold;
    u32 gate;
    u32 slot;
    u32 near;
    u32 cready;
    u32 abort_;
    u32 go, pad[3];
    u32 sync;
    u32x4 stage[16][8][64];
};

#define LAS __attribute__((address_space(3)))
__device__ __forceinline__ u32 lds_ld(u32* p) { return __builtin_amdgcn_readfirstlane(*(volatile LAS u32*)p); }
__device__ __forceinline__ void lds_st(u32* p, u32 v) { *(volatile LAS u32*)p = v; }

__device__ __forceinline__ void wait_until(u32 t) {
    for (;;) {
        const int rem = (int)(t - (u32)__builtin_amdgcn_s_memrealtime());
        if (rem <= 0) break;
        __builtin_amdgcn_s_sleep(1);
    }
}

__device__ __forceinline__ u32 link_shift(const char* ws, int g, u32 call, int lane) {
    const u64 v = __hip_atomic_load((const GAS u64*)(ws + GATE_OFF + g * FROW + 128 + (lane & (VOTES - 1)) * 8), __ATOMIC_RELAXED, __HIP_MEMORY_SCOPE_AGENT);
    const u32 s = (u32)(v >> 16) == call ? (u32)v & 0xffffu : 0u;
    u32 m = __builtin_amdgcn_readlane(s, 0);
#pragma unroll
    for (int i = 1; i < VOTES; ++i) m = min(m, (u32)__builtin_amdgcn_readlane(s, i));
    return m;
}

__device__ __forceinline__ u32 pack_bf16(float a, float b) {
    f32x2 v = {a, b};
    bf16x2 r = __builtin_convertvector(v, bf16x2);
    return __builtin_bit_cast(u32, r);
}
#define STAGE() __builtin_amdgcn_sched_barrier(0)
__device__ __forceinline__ void gelu8(f32x2 (&y)[4]) {
    const float c2 = -2.0f * 0.7978845608028654f * 1.4426950408889634f;
    const f32x2 k1 = {c2 * 0.044715f, c2 * 0.044715f}, k0 = {c2, c2}, one = {1.0f, 1.0f};
    f32x2 t[4];
#pragma unroll
    for (int i = 0; i < 4; ++i) t[i] = y[i] * y[i];
    STAGE();
#pragma unroll
    for (int i = 0; i < 4; ++i) t[i] = __builtin_elementwise_fma(t[i], k1, k0);
    STAGE();
#pragma unroll
    for (int i = 0; i < 4; ++i) t[i] = y[i] * t[i];
    STAGE();
#pragma unroll
    for (int i = 0; i < 4; ++i) t[i] = f32x2{__builtin_amdgcn_exp2f(t[i][0]), __builtin_amdgcn_exp2f(t[i][1])};
    STAGE();
#pragma unroll
    for (int i = 0; i < 4; ++i) t[i] = one + t[i];
    STAGE();
#pragma unroll
    for (int i = 0; i < 4; ++i) t[i] = f32x2{__builtin_amdgcn_rcpf(t[i][0]), __builtin_amdgcn_rcpf(t[i][1])};
    STAGE();
#pragma unroll
    for (int i = 0; i < 4; ++i) y[i] = y[i] * t[i];
    STAGE();
}
__device__ __forceinline__ float bf_lo(u32 v) { return __builtin_bit_cast(float, v << 16); }
__device__ __forceinline__ float bf_hi(u32 v) { return __builtin_bit_cast(float, v & 0xffff0000u); }

__device__ __forceinline__ u32 wchunk(int r) { return ((r & 3) * 4 + (r >> 2)) * 16; }

__device__ __forceinline__ void transpose8x8(u32x4 (&v)[8]) {
    u32x4 o[8];
#pragma unroll
    for (int c = 0; c < 4; ++c) {
#pragma unroll
        for (int r = 0; r < 4; ++r) {
            u32 a = v[2 * r][c], b = v[2 * r + 1][c];
            o[2 * c][r] = __builtin_amdgcn_perm(b, a, 0x05040100u);
            o[2 * c + 1][r] = __builtin_amdgcn_perm(b, a, 0x07060302u);
        }
    }
#pragma unroll
    for (int i = 0; i < 8; ++i) v[i] = o[i];
}

__global__ void __launch_bounds__(NTHREADS, 2) fused_gemm_rs(Params p) {
    __shared__ Shared sh;
    const int lane = threadIdx.x & 63;
    const int wave = __builtin_amdgcn_readfirstlane(threadIdx.x >> 6);
    const int grp = blockIdx.x & 7;
    const int tau = blockIdx.x >> 3;
    const int rank = p.rank;
    const int peer = (rank ^ grp) & 7;
    char* const myws = p.ws[0];
    char* const dws = p.ws[grp];
    const char* const px = p.x;
    const char* const pw_ = p.w;
    const int order0 = p.order[tau];
    asm volatile("" :: "s"(rank), "s"(myws), "s"(dws), "s"(px), "s"(pw_), "s"(order0));
    const u32 tstart = (u32)__builtin_amdgcn_s_memrealtime() + START_TICKS;

    if (wave >= 4) __builtin_amdgcn_s_setprio(3);
    char* const seqp = myws + SEQ_OFF + (size_t)(blockIdx.x * 8 + wave) * 4;
    if (threadIdx.x == 0) { sh.ready = 0; sh.hold = 0; sh.gate = 0; sh.near = 0; sh.cready = 0; sh.abort_ = 0; sh.sync = 0; sh.go = 0; }
    asm volatile("s_waitcnt lgkmcnt(0)" ::: "memory");

    if (wave == 4) {
        const int pw = tau;
        const int pi = (peer >> 2) & 1, pj = (peer >> 1) & 1, pk = peer & 1;
        const int q = lane >> 4, r = lane & 15;
        const rsrc_t rsx = mk_rsrc(px, 4 * MIB), rsw = mk_rsrc(pw_, 4 * MIB);
        const u32 xv = (pi * 256 + r) * 1024 + pk * 512 + q * 16;
        const u32 wv = (pk * 256 + q * 8) * 8192 + pj * 4096 + wchunk(r);
        u32x4 va[8], vb[8];
        auto issue_loads = [&](int o) {
            const int ks = (o >> 3) & 7, idx = o & 7;
            if (o & 64) {
                const u32 so = ks * 32 * 8192 + idx * 512;
#pragma unroll
                for (int i = 0; i < 8; ++i) { va[i] = ld16(rsw, wv, so + i * 8192); vb[i] = ld16(rsw, wv + 256, so + i * 8192); }
            } else {
                const u32 so = ((rank ^ idx) & 7) * 512 * 1024 + ks * 64;
#pragma unroll
                for (int i = 0; i < 8; ++i) { va[i] = ld16(rsx, xv, so + i * 16384); vb[i] = ld16(rsx, xv, so + (i + 8) * 16384); }
            }
        };
        const bool eager = grp != 0 && pw < EAGER;
        const u32 seq_raw = ld_agent(seqp);
        if (eager) issue_loads(order0);
        __builtin_amdgcn_s_barrier();
        if (!eager) {
            wait_until(tstart - LAZY + (grp != 0 ? ((pw - EAGER) * DELTA_Q4) >> 5 : 0));
            issue_loads(order0);
        }
        const u32 call = __builtin_amdgcn_readfirstlane(seq_raw) + 1;
        if (lane == 0) st_agent(seqp, call);
        const rsrc_t rsd = mk_rsrc(dws, FLAG_OFF);
        const u32 dpar = (call & 1) * (u32)PAR_BYTES;
        auto clear = [&](int k) {
            const rsrc_t rsm = mk_rsrc(myws, FLAG_OFF);
            const u32x4 zero = {0, 0, 0, 0};
#pragma unroll
            for (int i = 0; i < 8; ++i)
                st16_sys(rsm, lane * 16 + (i & 3) * 1024, (u32)PAR_BYTES - dpar + (u32)CB_OFF + (blockIdx.x * 4 + k) * 8192 + (i >> 2) * 4096, zero);
        };
        u32 shift = 0;
        for (int u = pw; u < NUNITS; u += PUSHERS) {
            const int o = p.order[u];
            const bool isw = o & 64;
            const int ks = (o >> 3) & 7, idx = isw ? (o & 7) : ((o ^ grp) & 7);
            const int n = grp == 0 ? ks : 8 + ks * 7 + (grp - 1);
            if (isw) { transpose8x8(va); transpose8x8(vb); }
            const u32 d = dpar + (u32)(isw ? WB_OFF : XB_OFF) + idx * IDX_STRIDE + n * STEP_STRIDE;
            const u32 nominal = tstart + ((u * DELTA_Q4) >> 4);
            if (grp != 0) {
                if (u >= 8) {
                    for (int i = 0; i < 8; ++i) {
                        wait_until(nominal + shift - SHIFT_LEAD);
                        const u32 old = shift;
                        shift = link_shift(myws, grp, call, lane);
                        if (shift < old + 40) break;
                    }
                }
                wait_until(nominal + shift - HOLD_PRE);
                if (lane == 0) lds_st(&sh.hold, 1);
                wait_until(nominal + shift);
            }
#pragma unroll
            for (int i = 0; i < 8; ++i) st16_sys(rsd, lane * 16 + (i & 3) * 1024, d + (i >> 2) * 4096, va[i]);
#pragma unroll
            for (int i = 0; i < 8; ++i) st16_sys(rsd, lane * 16 + (i & 3) * 1024, d + 8192 + (i >> 2) * 4096, vb[i]);
            if (lane == 0) lds_st(&sh.hold, 0);
            if (grp == 0 && u + PUSHERS < NUNITS) {
                issue_loads(p.order[u + PUSHERS]);
                WAIT_VM(16);
            } else {
                WAIT_VM(0);
            }
            if (grp != 0) {
                const int over = (int)((u32)__builtin_amdgcn_s_memrealtime() - ACK_OK - nominal - shift);
                if (over > 0 && lane == 0)
                    __hip_atomic_fetch_max((GAS u64*)(myws + GATE_OFF + grp * FROW + 128 + (u & (VOTES - 1)) * 8), ((u64)call << 16) + min(shift + min(over, SHIFT_CAP), 4000u),
                                           __ATOMIC_RELAXED, __HIP_MEMORY_SCOPE_AGENT);
            }
            if (lane == 0) { st_sys(dws + (isw ? FW_OFF : FX_OFF) + idx * FROW + n * 4, call); lds_st(&sh.hold, 0); lds_st(&sh.go, 1); }
            if (grp != 0 && u + PUSHERS < NUNITS) issue_loads(p.order[u + PUSHERS]);
            clear(u / PUSHERS);
        }
        if (pw + 3 * PUSHERS >= NUNITS) clear(3);
        return;
    }

    __builtin_amdgcn_s_barrier();
    const u32 call = __builtin_amdgcn_readfirstlane(ld_agent(seqp)) + 1;
    if (lane == 0) st_agent(seqp, call);
    const int rb = tau & 1, cb = tau >> 1;
    const int quad = grp >> 1;
    const int rtau = (tau + RSKEW * ((grp >> 2) * 2 + (grp & 1))) & 31;

    if (wave == 5) {
        const u64 t0 = __builtin_amdgcn_s_memrealtime();
        bool dead = false;
        u32 polls = 0;
        const char* priv = myws + PRIV_OFF + (size_t)blockIdx.x * FROW + (lane & 7) * 4;
        auto expired = [&](u32 pv) -> bool {
            if (__builtin_amdgcn_readlane(pv, 4) == call) return true;
            if ((++polls & 255) == 0 && __builtin_amdgcn_s_memrealtime() - t0 > DEADLINE_TICKS) {
                for (int i = 0; i < 4; ++i) st_agent(myws + PRIV_OFF + (size_t)(i * 64 + lane) * FROW + 16, call);
                return true;
            }
            return false;
        };
        bool master = grp == 0 && tau < 16;
        const char* frow = myws + FX_OFF + tau * FROW + lane * 4;
        const int cons = tau < 8 ? (lane & 31) * 8 + tau : (tau - 8) * 32 + (lane & 31);
        char* const pub_w = myws + PRIV_OFF + (size_t)cons * FROW + (tau >> 3) * 4;
        const bool skip = tau < 8 && (lane < 8 ? tau == 0 : (lane - 8) % 7 == tau - 1);
        u32 pubd = 0, seen = 0;
        for (;;) {
            u32 a = 0;
            if (master) a = ld_sys(frow);
            const u32 pv = ld_sys(priv);
            if (master) {
                const u64 m = ~__ballot(a >= call || skip);
                const u32 c = m ? (u32)__builtin_ctzll(m) : 64u;
                if (c != pubd) {
                    pubd = c;
                    if (lane < 32) st_agent(pub_w, call * 128 + c);
                    if (c == 64) master = false;
                }
            }
            const int c0 = (int)(__builtin_amdgcn_readlane(pv, 0) - call * 128), c1 = (int)(__builtin_amdgcn_readlane(pv, 1) - call * 128);
            const int c = min(c0, c1);
            if (c > (int)seen) {
                seen = c;
                if (lane == 0) lds_st(&sh.ready, seen);
            }
            if (seen == 64 && !master) break;
            if (expired(pv)) { dead = true; break; }
            if (master) __builtin_amdgcn_s_sleep(MASTER_SLEEP); else __builtin_amdgcn_s_sleep(POLL_SLEEP);
        }
        if (dead && lane == 0) { lds_st(&sh.abort_, 1); lds_st(&sh.ready, 64); }
        while (!lds_ld(&sh.near)) __builtin_amdgcn_s_sleep(2);
        wait_until(tstart + ((NUNITS * DELTA_Q4) >> 4) - 250);
        const u32 shift = grp != 0 ? link_shift(myws, grp, call, lane) : 0;
        if (lane == 0) {
            const u64 now = __builtin_amdgcn_s_memrealtime();
            u64 slot = now;
            if (grp != 0) {
                u64 first = now + (u64)(long long)(int)(tstart + ((NUNITS * DELTA_Q4) >> 4) + shift - (u32)now);
                u64 lo = now + NEAR_TICKS;
                if ((long long)(first - lo) > 0) lo = first;
                GAS u64* clk = (GAS u64*)(myws + GATE_OFF + grp * FROW);
                __hip_atomic_fetch_max(clk, lo << 3, __ATOMIC_RELAXED, __HIP_MEMORY_SCOPE_AGENT);
                slot = __hip_atomic_fetch_add(clk, (u64)DELTA_C, __ATOMIC_RELAXED, __HIP_MEMORY_SCOPE_AGENT) >> 3;
                if (slot - now > 100000ull) slot = now;
            }
            lds_st(&sh.slot, (u32)slot);
            lds_st(&sh.gate, 1);
        }
        for (int h = 0; h < 2; ++h) {
            wait_until(lds_ld(&sh.slot) + (((2 * h + 1) * DELTA_C) >> 5) - ANN);
            if (lane == 0) st_sys(dws + PRIV_OFF + (size_t)(((tau - RSKEW * ((rank >> 2) * 2 + h)) & 31) * 8 + (rank >> 1) * 2 + h) * FROW + 8 + (rank & 1) * 4, call);
        }
        while (!dead) {
            const u32 pv = ld_sys(priv);
            const bool got = __builtin_amdgcn_readlane(pv, 2) >= call && __builtin_amdgcn_readlane(pv, 3) >= call;
            if (lane == 0) lds_st(&sh.cready, got);
            if (got) break;
            dead = expired(pv);
            __builtin_amdgcn_s_sleep(CPOLL_SLEEP);
        }
        if (dead && lane == 0) lds_st(&sh.abort_, 1);
        if (lane == 0) lds_st(&sh.cready, 1);
        return;
    }

    const int wr = wave >> 1, wc = wave & 1;
    const rsrc_t rsm = mk_rsrc(myws, FLAG_OFF);
    const u32 par = (call & 1) * (u32)PAR_BYTES;
    const u32 xa = par + (u32)XB_OFF + grp * IDX_STRIDE + (rb * 8 + wr * 4) * 1024;
    const u32 wa = par + (u32)WB_OFF + (cb >> 1) * IDX_STRIDE + ((cb & 1) * 8 + wc * 4) * 1024;
    const u32 loff = lane * 16;
    u32x2 st = {0, 0};
    auto peek = [&]() { st = *(volatile LAS u32x2*)&sh.ready; };
    auto wait_step = [&](int n) {
        for (;;) {
            const int rdy = __builtin_amdgcn_readfirstlane(st[0]);
            const u32 hold = __builtin_amdgcn_readfirstlane(st[1]);
            if (rdy > n && !hold) break;
            __builtin_amdgcn_s_sleep(2);
            peek();
        }
        asm volatile("" ::: "memory");
    };
    const int orow = (quad >> 1) * 256 + rb * 128 + (grp & 1) * 64 + wave * 16 + (lane & 15);
    const u32 cva = (u32)CB_OFF + ((quad * 2 * 32 + rtau) * 4 + (grp & 1) * 2) * 8192 + wave * 2048;
    while (!lds_ld(&sh.go)) __builtin_amdgcn_s_sleep(4);
#pragma nounroll
    for (int phase = 0; phase < 2; ++phase) {
    if ((phase == 0) == (tau >= DIAG_LATE)) {
        if (phase) wait_until(lds_ld(&sh.slot) + (DELTA_C >> 3) + 60);
        const rsrc_t rsx = mk_rsrc(px, 4 * MIB), rsw = mk_rsrc(pw_, 4 * MIB);
        const u32 xv = (rank * 512 + orow) * 1024 + (lane >> 4) * 16;
        const u32 wv = (lane >> 4) * 8 * 8192 + (quad & 1) * 4096 + (rtau >> 1) * 256 + wchunk(lane & 15);
        auto meet = [&](u32 target) {
            asm volatile("s_waitcnt lgkmcnt(0)" ::: "memory");
            if (lane == 0) __hip_atomic_fetch_add(&sh.sync, 1u, __ATOMIC_RELAXED, __HIP_MEMORY_SCOPE_WORKGROUP);
            while (lds_ld(&sh.sync) < target) __builtin_amdgcn_s_sleep(1);
            asm volatile("" ::: "memory");
        };
        u32x4 wb[4][8], xb[16];
#pragma unroll
        for (int t = 0; t < 4; ++t) {
            peek();
            wait_step(-1);
#pragma unroll
            for (int i = 0; i < 8; ++i) wb[t][i] = ld16(rsw, wv, ((wave * 4 + t) * 32 + i) * 8192);
#pragma unroll
            for (int i = 0; i < 2; ++i) xb[t * 2 + i] = ld16(rsx, xv, (t * 2 + i) * 64);
            if (DIAG_GAP) __builtin_amdgcn_s_sleep(DIAG_GAP);
        }
#pragma unroll
        for (int t = 0; t < 4; ++t) {
            transpose8x8(wb[t]);
#pragma unroll
            for (int i = 0; i < 8; ++i) sh.stage[wave * 4 + t][i][lane] = wb[t][i];
            __builtin_amdgcn_sched_barrier(0);
            if (t == 0) {
#pragma unroll
                for (int i = 8; i < 16; ++i) xb[i] = ld16(rsx, xv, i * 64);
            }
        }
        meet(4);
        LAS char* sb[2] = {(LAS char*)&sh.stage[0][0][lane], (LAS char*)&sh.stage[8][0][lane]};
        asm volatile("" : "+v"(sb[0]), "+v"(sb[1]));
        f32x4 d[8];
#pragma unroll
        for (int i = 0; i < 8; ++i) d[i] = f32x4{0.f, 0.f, 0.f, 0.f};
#pragma unroll
        for (int k = 0; k < 17; ++k) {
            if (k < 16) {
#pragma unroll
                for (int i = 0; i < 8; ++i) wb[k & 1][i] = *(LAS u32x4*)(sb[k >> 3] + (k & 7) * 8192 + i * 1024);
            }
            __builtin_amdgcn_sched_barrier(0);
            if (k > 0) {
#pragma unroll
                for (int i = 0; i < 8; ++i)
                    d[i] = __builtin_amdgcn_mfma_f32_16x16x32_bf16(__builtin_bit_cast(bf16x8, wb[(k - 1) & 1][i]), __builtin_bit_cast(bf16x8, xb[k - 1]), d[i], 0, 0, 0);
            }
            __builtin_amdgcn_sched_barrier(0);
        }
        meet(8);
#pragma unroll
        for (int i = 0; i < 8; ++i) {
            const int r = i >> 1, c = (i & 1) * 4;
            sh.stage[wave][i][lane] = __builtin_bit_cast(u32x4, f32x4{d[c][r], d[c + 1][r], d[c + 2][r], d[c + 3][r]});
        }
    }
    if (phase != 0 || DRY != 0) {
        const rsrc_t rsc = mk_rsrc(myws, phase ? FLAG_OFF : 0), rso = mk_rsrc(p.out, phase ? 4 * MIB : 0);
        f32x4 d[8];
#pragma unroll
        for (int i = 0; i < 8; ++i) d[i] = __builtin_bit_cast(f32x4, sh.stage[wave][i][lane]);
#pragma unroll
        for (int i = 0; i < 8; ++i) asm volatile("" : "+v"(d[i]));
        while (phase && !lds_ld(&sh.cready)) __builtin_amdgcn_s_sleep(1);
        asm volatile("" ::: "memory");
        bool bad = phase && lds_ld(&sh.abort_) != 0;
        u32x4 v[2][2][2], w2[2][2][2];
        auto fetch = [&](u32x4 (&q)[2][2][2]) {
#pragma unroll
            for (int s = 0; s < 2; ++s)
#pragma unroll
                for (int c = 0; c < 2; ++c)
#pragma unroll
                    for (int h = 0; h < 2; ++h)
                        q[s][c][h] = ld16<AUX_SYS>(rsc, loff + h * 1024, par + cva + (s * 32 * 4 + c) * 8192);
        };
        auto complete = [&](u32x4 (&q)[2][2][2]) -> bool {
            u32 mn = ~0u;
#pragma unroll
            for (int i = 0; i < 8; ++i)
#pragma unroll
                for (int e = 0; e < 4; ++e) mn = min(mn, q[i >> 2][(i >> 1) & 1][i & 1][e]);
            return __ballot(mn == 0) == 0;
        };
        fetch(v);
        STAGE();
        __builtin_amdgcn_s_sleep(STAG);
        asm volatile("" ::: "memory");
        fetch(w2);
        STAGE();
        if (!complete(v) && phase && !bad) {
            for (;;) {
                if (complete(w2)) break;
                if ((u32)__builtin_amdgcn_s_memrealtime() - tstart > 2 * (u32)DEADLINE_TICKS) { bad = true; break; }
                fetch(w2);
            }
#pragma unroll
            for (int i = 0; i < 8; ++i) v[i >> 2][(i >> 1) & 1][i & 1] = w2[i >> 2][(i >> 1) & 1][i & 1];
        }
        const int ocol = (quad & 1) * 2048 + (rtau >> 1) * 128 + (lane >> 4) * 8;
        const u32 o = orow * 8192 + ocol * 2;
        if (bad) {
            const u32x4 nanv = {0x7fc07fc0u, 0x7fc07fc0u, 0x7fc07fc0u, 0x7fc07fc0u};
#pragma unroll
            for (int r = 0; r < 4; ++r) __builtin_amdgcn_raw_buffer_store_b128(nanv, rso, o + r * 64, 0, 0);
            return;
        }
#pragma unroll
        for (int r = 0; r < 4; ++r) {
            const int h = r >> 1, e = (r & 1) * 2;
            f32x2 y[4], yb[4];
#pragma unroll
            for (int i = 0; i < 4; ++i) {
                const u32 a = v[0][i >> 1][h][e + (i & 1)], b = v[1][i >> 1][h][e + (i & 1)];
                y[i] = f32x2{bf_lo(a), bf_hi(a)};
                yb[i] = f32x2{bf_lo(b), bf_hi(b)};
            }
            STAGE();
#pragma unroll
            for (int i = 0; i < 4; ++i) y[i] += yb[i];
            STAGE();
#pragma unroll
            for (int i = 0; i < 4; ++i) y[i] += f32x2{d[r * 2 + (i >> 1)][(i & 1) * 2], d[r * 2 + (i >> 1)][(i & 1) * 2 + 1]};
            STAGE();
            gelu8(y);
            u32x4 res;
#pragma unroll
            for (int i = 0; i < 4; ++i) res[i] = __builtin_bit_cast(u32, __builtin_convertvector(y[i], bf16x2));
            __builtin_amdgcn_raw_buffer_store_b128(res, rso, o + r * 64, 0, OUT_AUX);
        }
        STAGE();
#pragma unroll
        for (int i = 0; i < 8; ++i) asm volatile("" :: "v"(w2[i >> 2][(i >> 1) & 1][i & 1]));
    }
    if (phase) break;
    f32x4 acc[4][4];
#pragma unroll
    for (int a = 0; a < 4; ++a)
#pragma unroll
        for (int b = 0; b < 4; ++b) acc[a][b] = f32x4{0.f, 0.f, 0.f, 0.f};
    bf16x8 fxr[4][4], fwr[4][4];
    int n = grp == 0 ? 8 : 0, nskip = grp == 0 ? 99 : 7 + grp;
    auto load_step = [&](bf16x8 (&fx)[4], bf16x8 (&fw)[4]) {
        wait_step(n);
#pragma unroll
        for (int i = 0; i < 4; ++i) fx[i] = __builtin_bit_cast(bf16x8, ld16(rsm, loff + i * 1024, xa + n * STEP_STRIDE));
#pragma unroll
        for (int i = 0; i < 4; ++i) fw[i] = __builtin_bit_cast(bf16x8, ld16(rsm, loff + i * 1024, wa + n * STEP_STRIDE));
        ++n;
        if (n == nskip) { ++n; nskip += 7; }
    };
    auto mma_step = [&](bf16x8 (&fx)[4], bf16x8 (&fw)[4]) {
#pragma unroll
        for (int a = 0; a < 4; ++a)
#pragma unroll
            for (int b = 0; b < 4; ++b)
                acc[a][b] = __builtin_amdgcn_mfma_f32_16x16x32_bf16(fw[b], fx[a], acc[a][b], 0, 0, 0);
    };
#pragma unroll
    for (int s = 0; s < 3; ++s) load_step(fxr[s], fwr[s]);
    for (int m = 0; m < 52; m += 4) {
#pragma unroll
        for (int s = 0; s < 4; ++s) {
            load_step(fxr[(s + 3) & 3], fwr[(s + 3) & 3]);
            peek();
            __builtin_amdgcn_sched_barrier(0);
            mma_step(fxr[s], fwr[s]);
        }
    }
    load_step(fxr[3], fwr[3]);
    if (wave == 0 && lane == 0) lds_st(&sh.near, 1);
    __builtin_amdgcn_sched_barrier(0);
#pragma unroll
    for (int s = 0; s < 4; ++s) mma_step(fxr[s], fwr[s]);

    {
        u32x4 pk[8];
#pragma unroll
        for (int mt = 0; mt < 4; ++mt)
#pragma unroll
            for (int h = 0; h < 2; ++h) {
                pk[mt * 2 + h][0] = pack_bf16(acc[mt][0][2 * h], acc[mt][1][2 * h]);
                pk[mt * 2 + h][1] = pack_bf16(acc[mt][2][2 * h], acc[mt][3][2 * h]);
                pk[mt * 2 + h][2] = pack_bf16(acc[mt][0][2 * h + 1], acc[mt][1][2 * h + 1]);
                pk[mt * 2 + h][3] = pack_bf16(acc[mt][2][2 * h + 1], acc[mt][3][2 * h + 1]);
            }
#pragma unroll
        for (int i = 0; i < 8; ++i)
#pragma unroll
            for (int e = 0; e < 4; ++e) pk[i][e] = pk[i][e] ? pk[i][e] : 0x8000u;
        while (!lds_ld(&sh.gate)) __builtin_amdgcn_s_sleep(2);
        if (lds_ld(&sh.abort_)) {
#pragma unroll
            for (int i = 0; i < 8; ++i) pk[i] = u32x4{0x7fc07fc0u, 0x7fc07fc0u, 0x7fc07fc0u, 0x7fc07fc0u};
        }
        wait_until(lds_ld(&sh.slot) + ((wave * DELTA_C) >> 5));
        const rsrc_t rsd = mk_rsrc(dws, FLAG_OFF);
        const u32 dst = par + (u32)CB_OFF + ((rank * 32 + tau) * 4 + wave) * 8192;
#pragma unroll
        for (int i = 0; i < 8; ++i) st16_sys(rsd, loff + (i & 3) * 1024, dst + (i >> 2) * 4096, pk[i]);
    }
    }
}

extern "C" void kernel_launch(int rank, int nranks, ncclComm_t comm,
                              hipStream_t stream,
                              void* const* d_in, const int* in_sizes, int n_in,
                              void* d_out, int out_size,
                              const int* in_dtypes, int out_dtype,
                              void* const* d_out_all,
                              void* const* d_ws_all, size_t ws_bytes) {
    (void)comm; (void)in_dtypes; (void)out_dtype; (void)d_out_all;
    if (nranks != NR || (unsigned)rank >= (unsigned)NR || n_in != 2 || ws_bytes < WS_NEED ||
        in_sizes[0] != 4096 * 512 || in_sizes[1] != 512 * 4096 || out_size != 512 * 4096) {
        fprintf(stderr, "kernel_launch: unexpected problem shape\n");
        return;
    }
    Params p;
    p.x = (const char*)d_in[0];
    p.w = (const char*)d_in[1];
    p.out = (char*)d_out;
    for (int i = 0; i < NR; ++i) p.ws[i] = (char*)d_ws_all[rank ^ i];
    p.rank = rank;
    p.pad = 0;
    int n = 0;
    for (int key = 0; n < NUNITS; ++key)
        for (int ks = 0; ks < 8; ++ks) {
            if (ks * KEY_K == key)
                for (int i = 1; i < 8; ++i) p.order[n++] = (u32)(ks * 8 + i);
            for (int g = 0; g < 8; ++g)
                if (ks * KEY_K + g * KEY_G == key) p.order[n++] = (u32)(64 + ks * 8 + g);
        }
    hipLaunchKernelGGL(fused_gemm_rs, dim3(NWG), dim3(NTHREADS), 0, stream, p);
    HIP_CHECK(hipGetLastError());
}
```
